```python
import numpy as np

kernel_name = "gpu_a2a_gemm_m4096_k4096_n4096_e4m3e5m2_dqrelu"


ABSMAX_RTOL = 0.05

COMPUTE_DTYPE = "float32"
EPILOGUE = "dequant_relu"
IN_QUANT = None
OUT_QUANT = None
IN_FP8 = "e4m3e5m2"
IN_DTYPES = {"x": "e4m3", "w_mat": "e5m2",
             "scale_x": "float32", "scale_w": "float32"}
WORLD_SIZE = 8

def _snap_fp8(x, mant_bits, max_abs, min_normal):
    a = np.abs(x.astype(np.float64))
    _m, _e = np.frexp(a)
    q = np.where(a >= min_normal, np.ldexp(1.0, _e - 1 - mant_bits),
                 min_normal * 2.0 ** -mant_bits)
    snapped = np.minimum(np.round(a / q) * q, max_abs)
    return (np.sign(x) * snapped).astype(np.float32)

def setup_inputs():

    x = _snap_fp8(np.random.randn(4096, 4096).astype(np.float32), 3, 448.0, 2.0 ** -6)
    w_mat = _snap_fp8(np.random.randn(4096, 4096).astype(np.float32), 2, 57344.0, 2.0 ** -14)
    scale_x = (0.02 * (1.0 + np.random.rand(1))).astype(np.float32)
    scale_w = (0.02 * (1.0 + np.random.rand(1))).astype(np.float32)
    return {"x": x, "w_mat": w_mat, "scale_x": scale_x, "scale_w": scale_w}

def reference(x, w_mat, scale_x, scale_w):
    acc = (x @ w_mat).astype(np.float32)
    y = (acc * (scale_x * scale_w)).astype(np.float32)
    return np.maximum(y, 0.0).astype(np.float32)
```

```cpp
#include <hip/hip_runtime.h>
#include <rccl/rccl.h>
#include <cstdio>
#include <cstdint>

#define HIP_CHECK(expr) do { hipError_t _e = (expr); if (_e != hipSuccess) { \
    fprintf(stderr, "HIP error %s at %s:%d\n", hipGetErrorString(_e), __FILE__, __LINE__); \
    return; } } while (0)

typedef int v2i __attribute__((ext_vector_type(2)));
typedef int v4i __attribute__((ext_vector_type(4)));
typedef int v8i __attribute__((ext_vector_type(8)));
typedef float v4f __attribute__((ext_vector_type(4)));
typedef float v16f __attribute__((ext_vector_type(16)));
#define LDSP __attribute__((address_space(3)))

constexpr int NR = 8;
constexpr int MLOC = 512;
constexpr int KB = 512;
constexpr int NDIM = 4096;
constexpr int BM = 64, BN = 128, BK = 128;
constexpr int NTHREADS = 512;
constexpr int NWG = 256;
constexpr int NSTEP = 32;
constexpr int NS = 6;
constexpr int A_BYTES = BM * BK;
constexpr int B_BYTES = BK * BN;
constexpr int ST_BYTES = A_BYTES + B_BYTES;
constexpr int LDS_MISC = NS * ST_BYTES;
constexpr int LDS_BYTES = LDS_MISC + 256;

constexpr size_t WS_CTR = 0;
constexpr size_t WS_TMO = 64 * 1024;
constexpr size_t WS_FLAG = 128 * 1024;
constexpr size_t WS_STAGE = 1024 * 1024;
constexpr size_t BLK_BYTES = (size_t)MLOC * KB;
constexpr size_t WS_NEED = WS_STAGE + 2 * NR * BLK_BYTES;
constexpr unsigned long long DEADLINE_TICKS = 50000000ull;

struct Params {
  const uint8_t* x;
  const uint8_t* w;
  const float* sx;
  const float* sw;
  float* out;
  uint8_t* ws[NR];
  int rank;
};

#define WAIT_V(n) asm volatile("s_waitcnt vmcnt(" #n ")" ::: "memory")

__device__ __forceinline__ void glds16(const void* gsrc, unsigned lds_dst) {
  unsigned keep;
  asm volatile("s_mov_b32 %0, m0\n\ts_mov_b32 m0, %2\n\ts_nop 0\n\tglobal_load_lds_dwordx4 %1, off\n\ts_mov_b32 m0, %0"
               : "=&s"(keep) : "v"(gsrc), "s"(lds_dst) : "memory");
}
__device__ __forceinline__ void glds16_sc1(const void* gsrc, unsigned lds_dst) {
  unsigned keep;
  asm volatile("s_mov_b32 %0, m0\n\ts_mov_b32 m0, %2\n\ts_nop 0\n\tglobal_load_lds_dwordx4 %1, off sc1\n\ts_mov_b32 m0, %0"
               : "=&s"(keep) : "v"(gsrc), "s"(lds_dst) : "memory");
}

__global__ void __launch_bounds__(NTHREADS, 2) a2a_gemm_kernel(Params p) {
  extern __shared__ __attribute__((aligned(16))) uint8_t lds[];
  const int tid = threadIdx.x;
  const int lane = tid & 63;
  const int wave = __builtin_amdgcn_readfirstlane(tid >> 6);
  const int bid = blockIdx.x;
  const int rank = p.rank;
  uint8_t* const myws = p.ws[rank];

  unsigned* ctr = (unsigned*)(myws + WS_CTR + (size_t)bid * 128);
  const unsigned call = __builtin_amdgcn_readfirstlane(
      __hip_atomic_load(ctr, __ATOMIC_RELAXED, __HIP_MEMORY_SCOPE_AGENT)) + 1u;
  const unsigned par = call & 1u;
  unsigned* tmo = (unsigned*)(myws + WS_TMO);
  volatile LDSP unsigned* s_bad = (volatile LDSP unsigned*)(lds + LDS_MISC);
  if (tid == 0) *s_bad = 0u;

  if (wave < 2) {
    const int job = bid + NWG * wave;
    const unsigned dst = (unsigned)(job >> 6);
    const int rg = job & 63;
    if (dst < (unsigned)NR && (int)dst != rank) {
      const int row = rg * 8 + (lane >> 3);
      const uint8_t* src = p.x + ((size_t)dst * MLOC + row) * KB + (lane & 7) * 16;
      v4i v[4];
#pragma unroll
      for (int c = 0; c < 4; ++c) v[c] = *(const v4i*)(src + c * 128);
      uint8_t* dptr = p.ws[dst] + WS_STAGE + ((size_t)par * NR + rank) * BLK_BYTES + (size_t)row * KB + (lane & 7) * 16;
      unsigned* flag = (unsigned*)(p.ws[dst] + WS_FLAG) + rank * 64 + rg;
      asm volatile("" :: "v"(v[0]), "v"(v[1]), "v"(v[2]), "v"(v[3]));
#pragma unroll
      for (int c = 0; c < 4; ++c) {
        asm volatile("global_store_dwordx4 %0, %1, off sc0 sc1\n\ts_nop 1" :: "v"(dptr + c * 128), "v"(v[c]) : "memory");
      }
      const unsigned fbase = (call - 1u) * 4u;
      WAIT_V(3);
      if (lane == 0) __hip_atomic_store(flag, fbase + 1u, __ATOMIC_RELAXED, __HIP_MEMORY_SCOPE_SYSTEM);
      WAIT_V(3);
      if (lane == 0) __hip_atomic_store(flag, fbase + 2u, __ATOMIC_RELAXED, __HIP_MEMORY_SCOPE_SYSTEM);
      WAIT_V(3);
      if (lane == 0) __hip_atomic_store(flag, fbase + 3u, __ATOMIC_RELAXED, __HIP_MEMORY_SCOPE_SYSTEM);
      WAIT_V(3);
      if (lane == 0) __hip_atomic_store(flag, fbase + 4u, __ATOMIC_RELAXED, __HIP_MEMORY_SCOPE_SYSTEM);
      WAIT_V(0);
    }
  }

  const int g = bid & 7, idx = bid >> 3;
  const int nb = g * 4 + (idx & 3);
  const int rb = idx >> 2;

  const int prow = lane >> 3;
  const int a_lrow = wave * 8 + prow;
  const int a_c16 = (lane & 7) ^ ((a_lrow >> 1) & 7);
  const size_t a_off = (size_t)(rb * BM + a_lrow) * KB + a_c16 * 16;
  const int b_c16 = (lane & 7) ^ (((prow >> 1) & 3) << 1);
  const size_t b_off0 = (size_t)(wave * 16 + prow) * NDIM + nb * BN + b_c16 * 16;
  const size_t b_off1 = b_off0 + (size_t)8 * NDIM;
  const uint8_t* a_local = p.x + (size_t)rank * MLOC * KB;
  const uint8_t* a_stage = myws + WS_STAGE + (size_t)par * NR * BLK_BYTES;
  const unsigned* myflags = (const unsigned*)(myws + WS_FLAG);
  const unsigned need_base = (call - 1u) * 4u;
  unsigned fcache = 0;
  bool bad = false;
  const unsigned lds_base = (unsigned)(uintptr_t)(LDSP uint8_t*)lds;

  auto issue = [&](int s) {
    int src, c;
    if (s < 4) { src = rank; c = s; }
    else { const int t = s - 4; c = t / 7; src = (rank + 1 + (t - c * 7)) & 7; }
    const uint8_t* ab;
    if (s < 4) ab = a_local;
    else {
      ab = a_stage + (size_t)src * BLK_BYTES;
      const unsigned need = need_base + (unsigned)c + 1u;
      if (!bad && (unsigned)__builtin_amdgcn_readlane(fcache, src) < need) {
        const unsigned long long t0 = __builtin_amdgcn_s_memrealtime();
        for (;;) {
          if (lane < 8) fcache = __hip_atomic_load(myflags + lane * 64 + rb * 8 + wave, __ATOMIC_RELAXED, __HIP_MEMORY_SCOPE_SYSTEM);
          if ((unsigned)__builtin_amdgcn_readlane(fcache, src) >= need) break;
          if (__hip_atomic_load(tmo, __ATOMIC_RELAXED, __HIP_MEMORY_SCOPE_AGENT) == call ||
              __builtin_amdgcn_s_memrealtime() - t0 > DEADLINE_TICKS) {
            if (lane == 0) __hip_atomic_store(tmo, call, __ATOMIC_RELAXED, __HIP_MEMORY_SCOPE_AGENT);
            bad = true;
            break;
          }
          __builtin_amdgcn_s_sleep(1);
        }
      }
    }
    const unsigned sbase = lds_base + (unsigned)((s % NS) * ST_BYTES);
    glds16_sc1(ab + a_off + c * 128, sbase + wave * 1024);
    const uint8_t* wb = p.w + (size_t)(src * KB + c * 128) * NDIM;
    glds16(wb + b_off0, sbase + A_BYTES + wave * 2048);
    glds16(wb + b_off1, sbase + A_BYTES + wave * 2048 + 1024);
  };

  const int kk = wave >> 2, wm = (wave >> 1) & 1, wn = wave & 1;
  const int xrow = wm * 32 + (lane & 31);
  const int xc = kk * 4 + (lane >> 5) * 2;
  const int xsw = (xrow >> 1) & 7;
  const int x_addr0 = xrow * 128 + ((xc ^ xsw) * 16);
  const int x_addr1 = xrow * 128 + (((xc + 1) ^ xsw) * 16);
  const int q = (lane & 15) >> 1, pp = lane & 1;
  const int wsw = ((q >> 1) & 3) << 1;
  const int wkrow = kk * 64 + (lane >> 5) * 32 + q;
  const int wch = wn * 4 + ((lane >> 4) & 1);
  const int w_addr_t0 = A_BYTES + wkrow * 128 + ((wch ^ wsw) * 16) + pp * 8;
  const int w_addr_t1 = A_BYTES + wkrow * 128 + (((wch + 2) ^ wsw) * 16) + pp * 8;

  v16f acc0 = {}, acc1 = {};

#pragma unroll 1
  for (int s = 0; s < NS - 1; ++s) issue(s);

#pragma unroll 1
  for (int it = 0; it < NSTEP; ++it) {
    if (it <= NSTEP - NS + 1) { WAIT_V(12); }
    else if (it == NSTEP - 4) { WAIT_V(9); }
    else if (it == NSTEP - 3) { WAIT_V(6); }
    else if (it == NSTEP - 2) { WAIT_V(3); }
    else { WAIT_V(0); }
    __builtin_amdgcn_s_barrier();
    if (it + NS - 1 < NSTEP) issue(it + NS - 1);
    const uint8_t* sb = lds + (it % NS) * ST_BYTES;
    v8i xa, w0, w1;
    {
      v4i t0 = *(const LDSP v4i*)(LDSP uint8_t*)(sb + x_addr0);
      v4i t1 = *(const LDSP v4i*)(LDSP uint8_t*)(sb + x_addr1);
      xa = __builtin_shufflevector(t0, t1, 0, 1, 2, 3, 4, 5, 6, 7);
    }
#pragma unroll
    for (int r = 0; r < 4; ++r) {
      v2i a = __builtin_amdgcn_ds_read_tr8_b64_v2i32((LDSP v2i*)(LDSP uint8_t*)(sb + w_addr_t0 + r * 1024));
      v2i b = __builtin_amdgcn_ds_read_tr8_b64_v2i32((LDSP v2i*)(LDSP uint8_t*)(sb + w_addr_t1 + r * 1024));
      w0[2 * r] = a[0]; w0[2 * r + 1] = a[1];
      w1[2 * r] = b[0]; w1[2 * r + 1] = b[1];
    }
    acc0 = __builtin_amdgcn_mfma_scale_f32_32x32x64_f8f6f4(w0, xa, acc0, 1, 0, 0, 0x7F7F7F7F, 0, 0x7F7F7F7F);
    acc1 = __builtin_amdgcn_mfma_scale_f32_32x32x64_f8f6f4(w1, xa, acc1, 1, 0, 0, 0x7F7F7F7F, 0, 0x7F7F7F7F);
  }

  if (bad && lane == 0) *s_bad = 1u;
  asm volatile("s_waitcnt lgkmcnt(0)" ::: "memory");
  __builtin_amdgcn_s_barrier();
  {
    LDSP v4f* xb = (LDSP v4f*)(lds + (size_t)(wave ^ 4) * 4096);
    const v16f give = kk == 0 ? acc1 : acc0;
#pragma unroll
    for (int j = 0; j < 4; ++j) {
      v4f t = {give[4 * j], give[4 * j + 1], give[4 * j + 2], give[4 * j + 3]};
      xb[j * 64 + lane] = t;
    }
  }
  asm volatile("s_waitcnt lgkmcnt(0)" ::: "memory");
  __builtin_amdgcn_s_barrier();
  const bool anybad = (*s_bad != 0u) || (__hip_atomic_load(tmo, __ATOMIC_RELAXED, __HIP_MEMORY_SCOPE_AGENT) == call);
  {
    const float sc = p.sx[0] * p.sw[0];
    const LDSP v4f* ib = (const LDSP v4f*)(lds + (size_t)wave * 4096);
    const v16f keep = kk == 0 ? acc0 : acc1;
    const int m = rb * BM + wm * 32 + (lane & 31);
    const int n0 = nb * BN + wn * 64 + kk * 32 + (lane >> 5) * 4;
    float* op = p.out + (size_t)m * NDIM + n0;
#pragma unroll
    for (int j = 0; j < 4; ++j) {
      v4f t = ib[j * 64 + lane];
      v4f o;
#pragma unroll
      for (int e = 0; e < 4; ++e) {
        float y = (keep[4 * j + e] + t[e]) * sc;
        y = fmaxf(y, 0.0f);
        o[e] = anybad ? __builtin_nanf("") : y;
      }
      *(v4f*)(op + j * 8) = o;
    }
  }
  if (tid == 0) __hip_atomic_store(ctr, call, __ATOMIC_RELAXED, __HIP_MEMORY_SCOPE_AGENT);
}

extern "C" void kernel_launch(int rank, int nranks, ncclComm_t comm,
                              hipStream_t stream,
                              void* const* d_in, const int* in_sizes, int n_in,
                              void* d_out, int out_size,
                              const int* in_dtypes, int out_dtype,
                              void* const* d_out_all,
                              void* const* d_ws_all, size_t ws_bytes) {
  (void)comm; (void)in_sizes; (void)n_in; (void)out_size; (void)in_dtypes; (void)out_dtype; (void)d_out_all;
  if (nranks != NR || ws_bytes < WS_NEED) { fprintf(stderr, "unexpected config\n"); return; }
  static bool attr_done = false;
  if (!attr_done) {
    HIP_CHECK(hipFuncSetAttribute((const void*)a2a_gemm_kernel, hipFuncAttributeMaxDynamicSharedMemorySize, LDS_BYTES));
    attr_done = true;
  }
  Params p;
  p.x = (const uint8_t*)d_in[0];
  p.w = (const uint8_t*)d_in[1];
  p.sx = (const float*)d_in[2];
  p.sw = (const float*)d_in[3];
  p.out = (float*)d_out;
  for (int i = 0; i < NR; ++i) p.ws[i] = (uint8_t*)d_ws_all[i];
  p.rank = rank;
  a2a_gemm_kernel<<<NWG, NTHREADS, LDS_BYTES, stream>>>(p);
  HIP_CHECK(hipGetLastError());
}
```

```cpp
#include <hip/hip_runtime.h>
#include <rccl/rccl.h>
#include <cstdio>
#include <cstdint>

#define HIP_CHECK(expr) do { hipError_t _e = (expr); if (_e != hipSuccess) { \
    fprintf(stderr, "HIP error %s at %s:%d\n", hipGetErrorString(_e), __FILE__, __LINE__); \
    return; } } while (0)

typedef int v2i __attribute__((ext_vector_type(2)));
typedef int v4i __attribute__((ext_vector_type(4)));
typedef int v8i __attribute__((ext_vector_type(8)));
typedef float v4f __attribute__((ext_vector_type(4)));
typedef float v16f __attribute__((ext_vector_type(16)));
#define LDSP __attribute__((address_space(3)))

constexpr int NR = 8;
constexpr int MLOC = 512;
constexpr int KB = 512;
constexpr int NDIM = 4096;
constexpr int BM = 64, BN = 128, BK = 128;
constexpr int NWAVES = 11;
constexpr int NTHREADS = NWAVES * 64;
constexpr int NWG = 256;
constexpr int NSTEP = 32;
constexpr int NA = 5, NB = 7;
constexpr int LA = 3, LB = 5;
constexpr int A_BYTES = BM * BK;
constexpr int B_BYTES = BK * BN;
constexpr int LDS_A = 0;
constexpr int LDS_B = NA * A_BYTES;
constexpr int LDS_SYNC = LDS_B + NB * B_BYTES;
constexpr int LDS_SCR = LDS_SYNC + 64;
constexpr int LDS_TRC = LDS_SCR + 256;
constexpr int LDS_BYTES = LDS_TRC + 2048;

constexpr size_t WS_CTR = 0;
constexpr size_t WS_TMO = 64 * 1024;
constexpr size_t WS_FLAG = 128 * 1024;
constexpr size_t WS_STAGE = 1024 * 1024;
constexpr size_t BLK_BYTES = (size_t)MLOC * KB;
constexpr size_t WS_NEED = WS_STAGE + 2 * NR * BLK_BYTES;
constexpr unsigned long long DEADLINE_TICKS = 50000000ull;

struct Params {
  const uint8_t* x;
  const uint8_t* w;
  const float* sx;
  const float* sw;
  float* out;
  uint8_t* ws[NR];
  int rank;
};

#define WAIT_V(n) asm volatile("s_waitcnt vmcnt(" #n ")" ::: "memory")
#define WAIT_VN(n) asm volatile("s_waitcnt vmcnt(%0)" :: "n"(n) : "memory")
#define WAIT_L0() asm volatile("s_waitcnt lgkmcnt(0)" ::: "memory")

__device__ __forceinline__ void glds16(const void* gsrc, unsigned lds_dst) {
  unsigned keep;
  asm volatile("s_mov_b32 %0, m0\n\ts_mov_b32 m0, %2\n\ts_nop 0\n\tglobal_load_lds_dwordx4 %1, off\n\ts_mov_b32 m0, %0"
               : "=&s"(keep) : "v"(gsrc), "s"(lds_dst) : "memory");
}
__device__ __forceinline__ void glds16_sc1(const void* gsrc, unsigned lds_dst) {
  unsigned keep;
  asm volatile("s_mov_b32 %0, m0\n\ts_mov_b32 m0, %2\n\ts_nop 0\n\tglobal_load_lds_dwordx4 %1, off sc1\n\ts_mov_b32 m0, %0"
               : "=&s"(keep) : "v"(gsrc), "s"(lds_dst) : "memory");
}

__device__ __forceinline__ void glds4(const void* gsrc, unsigned lds_dst) {
  unsigned keep;
  asm volatile("s_mov_b32 %0, m0\n\ts_mov_b32 m0, %2\n\ts_nop 0\n\tglobal_load_lds_dword %1, off\n\ts_mov_b32 m0, %0"
               : "=&s"(keep) : "v"(gsrc), "s"(lds_dst) : "memory");
}

__device__ __forceinline__ void step_src(int s, int rank, int& src, int& c) {
  if (s < 4) { src = rank; c = s; }
  else { const int t = s - 4; c = t / 7; src = (rank + 1 + (t - c * 7)) & 7; }
}

__device__ __forceinline__ int lds_min4(unsigned a) {
  v4i v;
  asm volatile("ds_read_b128 %0, %1\n\ts_waitcnt lgkmcnt(0)" : "=v"(v) : "v"(a) : "memory");
  return __builtin_amdgcn_readfirstlane(min(min(v[0], v[1]), min(v[2], v[3])));
}
__device__ __forceinline__ int lds_min8(unsigned a) {
  v4i v, u;
  asm volatile("ds_read_b128 %0, %2\n\tds_read_b128 %1, %2 offset:16\n\ts_waitcnt lgkmcnt(0)" : "=&v"(v), "=&v"(u) : "v"(a) : "memory");
  return __builtin_amdgcn_readfirstlane(min(min(min(v[0], v[1]), min(v[2], v[3])), min(min(u[0], u[1]), min(u[2], u[3]))));
}

#define STAMP(i)
#define TRACE(i)
#define TRACE_DUMP()

struct Frag { v8i x[2]; v8i w[2][2]; };

__global__ void __launch_bounds__(NTHREADS, 1) a2a_gemm_kernel(Params p) {
  extern __shared__ __attribute__((aligned(16))) uint8_t lds[];
  STAMP(0);
  const int tid = threadIdx.x;
  const int lane = tid & 63;
  const int wave = __builtin_amdgcn_readfirstlane(tid >> 6);
  const int bid = blockIdx.x;
  const int rank = p.rank;
  uint8_t* const myws = p.ws[rank];
  volatile LDSP int* sync = (volatile LDSP int*)(lds + LDS_SYNC);
  const unsigned lds_base = (unsigned)(uintptr_t)(LDSP uint8_t*)lds;

  if (tid < 16) sync[tid] = (tid >= 5 && tid <= 7) ? 0x7fffffff : 0;
  WAIT_L0();
  __builtin_amdgcn_s_barrier();

  const int g = bid & 7, idx = bid >> 3;
  const int nb = g * 4 + (idx & 3);
  const int rb = idx >> 2;
  const int prow = lane >> 3;

  unsigned* ctr = (unsigned*)(myws + WS_CTR + (size_t)bid * 128);
  unsigned* tmo = (unsigned*)(myws + WS_TMO);

  if (wave >= 4 && wave < 8) {
    const int lw = wave - 4;
    const int b_c16 = (lane & 7) ^ (((prow >> 1) & 3) << 1);
    const size_t b_off = (size_t)(lw * 32 + prow) * NDIM + nb * BN + b_c16 * 16;
    int pub = 0, freec = 0;
#pragma unroll 1
    for (int s = 0; s < NSTEP; ++s) {
      if (freec < s - NB + 1) {
        freec = lds_min4(lds_base + LDS_SYNC + 32);
        if (freec < s - NB + 1) {
          WAIT_V(0);
          if (lane == 0) sync[lw] = s;
          pub = s;
          for (;;) {
            freec = lds_min4(lds_base + LDS_SYNC + 32);
            if (freec >= s - NB + 1) break;
            __builtin_amdgcn_s_sleep(1);
          }
        }
      }
      TRACE(s);
      int src, c; step_src(s, rank, src, c);
      const uint8_t* wb = p.w + (size_t)(src * KB + c * 128) * NDIM + b_off;
      const unsigned dst = lds_base + LDS_B + (unsigned)((s % NB) * B_BYTES) + lw * 4096;
#pragma unroll
      for (int j = 0; j < 4; ++j) glds16(wb + (size_t)j * 8 * NDIM, dst + j * 1024);
      if (s + 1 - LB > pub) {
        WAIT_VN(4 * LB);
        pub = s + 1 - LB;
        if (lane == 0) sync[lw] = pub;
      }
    }
#pragma unroll
    for (int r = LB - 1; r >= 0; --r) {
      WAIT_VN(4 * r);
      if (lane == 0 && NSTEP - r > pub) sync[lw] = NSTEP - r;
    }
    STAMP(1);
    TRACE_DUMP();
    return;
  }

  const unsigned call = __builtin_amdgcn_readfirstlane(
      __hip_atomic_load(ctr, __ATOMIC_RELAXED, __HIP_MEMORY_SCOPE_AGENT)) + 1u;
  const unsigned par = call & 1u;

  if (wave >= 9) {
    const int job = bid + NWG * (wave - 9);
    const unsigned dst = (unsigned)(job >> 6);
    const int rg = job & 63;
    if (dst < (unsigned)NR && (int)dst != rank) {
      const int row = rg * 8 + prow;
      const uint8_t* src = p.x + ((size_t)dst * MLOC + row) * KB + (lane & 7) * 16;
      v4i v[4];
#pragma unroll
      for (int c = 0; c < 4; ++c) v[c] = *(const v4i*)(src + c * 128);
      uint8_t* dptr = p.ws[dst] + WS_STAGE + ((size_t)par * NR + rank) * BLK_BYTES + (size_t)row * KB + (lane & 7) * 16;
      unsigned* flag = (unsigned*)(p.ws[dst] + WS_FLAG) + rank * 64 + rg;
      const unsigned fbase = (call - 1u) * 4u;
      asm volatile("" :: "v"(v[0]), "v"(v[1]), "v"(v[2]), "v"(v[3]));
#define PUSH_ST(c) asm volatile("global_store_dwordx4 %0, %1, off sc0 sc1\n\ts_nop 1" :: "v"(dptr + (c) * 128), "v"(v[c]) : "memory")
#define PUSH_FL(n) do { if (lane == 0) __hip_atomic_store(flag, fbase + (n), __ATOMIC_RELAXED, __HIP_MEMORY_SCOPE_SYSTEM); } while (0)
      PUSH_ST(0); PUSH_ST(1);
      WAIT_V(1); PUSH_FL(1u); PUSH_ST(2);
      WAIT_V(2); PUSH_FL(2u); PUSH_ST(3);
      WAIT_V(2); PUSH_FL(3u);
      WAIT_V(1); PUSH_FL(4u);
    }
    STAMP(1);
    TRACE_DUMP();
    return;
  }

  if (wave == 8) {
    const uint8_t* a_local = p.x + (size_t)rank * MLOC * KB;
    const uint8_t* a_stage = myws + WS_STAGE + (size_t)par * NR * BLK_BYTES;
    const unsigned* myflag = (const unsigned*)(myws + WS_FLAG) + (lane >> 3) * 64 + rb * 8 + (lane & 7);
    const unsigned need_base = (call - 1u) * 4u;
    const int sw_e = (lane & 7) ^ (prow >> 1);
    const int sw_o = (lane & 7) ^ (4 + (prow >> 1));
    const size_t a_off_e = (size_t)(rb * BM + prow) * KB + sw_e * 16;
    const size_t a_off_o = (size_t)(rb * BM + prow) * KB + sw_o * 16;
    unsigned fc = 0;
    bool bad = false;
    int pub = 0, freec = 0;
#pragma unroll 1
    for (int s = 0; s < NSTEP; ++s) {
      int src, c; step_src(s, rank, src, c);
      bool slot_ok = freec >= s - NA + 1;
      if (!slot_ok) { freec = lds_min4(lds_base + LDS_SYNC + 32); slot_ok = freec >= s - NA + 1; }
      const unsigned need = need_base + (unsigned)c + 1u;
      const unsigned long long want = 0xFFull << (src * 8);
      bool data_ok = (s < 4) || bad || ((__ballot(fc >= need) & want) == want);
      if (!slot_ok || !data_ok) {
        WAIT_V(0);
        if (lane == 0) sync[4] = s;
        pub = s;
        if (!data_ok) {
          const unsigned long long t0 = __builtin_amdgcn_s_memrealtime();
          for (;;) {
            fc = __hip_atomic_load(myflag, __ATOMIC_RELAXED, __HIP_MEMORY_SCOPE_SYSTEM);
            if ((__ballot(fc >= need) & want) == want) break;
            if (__hip_atomic_load(tmo, __ATOMIC_RELAXED, __HIP_MEMORY_SCOPE_AGENT) == call ||
                __builtin_amdgcn_s_memrealtime() - t0 > DEADLINE_TICKS) {
              if (lane == 0) { __hip_atomic_store(tmo, call, __ATOMIC_RELAXED, __HIP_MEMORY_SCOPE_AGENT); sync[12] = 1; }
              bad = true;
              break;
            }
            __builtin_amdgcn_s_sleep(1);
          }
        }
        while (freec < s - NA + 1) {
          freec = lds_min4(lds_base + LDS_SYNC + 32);
          if (freec >= s - NA + 1) break;
          __builtin_amdgcn_s_sleep(1);
        }
      }
      TRACE(s);
      const uint8_t* ab = (s < 4 ? a_local : a_stage + (size_t)src * BLK_BYTES) + c * 128;
      const unsigned dst = lds_base + LDS_A + (unsigned)((s % NA) * A_BYTES);
#pragma unroll
      for (int j = 0; j < 8; ++j) glds16_sc1(ab + ((j & 1) ? a_off_o : a_off_e) + (size_t)j * 8 * KB, dst + j * 1024);
      if (s + 1 - LA > pub) {
        WAIT_VN(8 * LA);
        pub = s + 1 - LA;
        if (lane == 0) sync[4] = pub;
      }
    }
#pragma unroll
    for (int r = LA - 1; r >= 0; --r) {
      WAIT_VN(8 * r);
      if (lane == 0 && NSTEP - r > pub) sync[4] = NSTEP - r;
    }
    STAMP(1);
    TRACE_DUMP();
    return;
  }

  __builtin_amdgcn_s_setprio(2);
  const int wm = wave >> 1, wn = wave & 1;
  const int xrow = wm * 32 + (lane & 31);
  const int xsw = (xrow >> 1) & 7;
  int x_addr[2][2];
#pragma unroll
  for (int kk = 0; kk < 2; ++kk)
#pragma unroll
    for (int h = 0; h < 2; ++h) x_addr[kk][h] = LDS_A + xrow * 128 + (((kk * 4 + (lane >> 5) * 2 + h) ^ xsw) * 16);
  const int q = (lane & 15) >> 1, pp = lane & 1;
  const int wsw = ((q >> 1) & 3) << 1;
  const int wkrow = (lane >> 5) * 32 + q;
  const int wch = wn * 4 + ((lane >> 4) & 1);
  int w_addr[2];
#pragma unroll
  for (int t = 0; t < 2; ++t) w_addr[t] = LDS_B + wkrow * 128 + (((wch + 2 * t) ^ wsw) * 16) + pp * 8;

  auto load_frag = [&](Frag& f, int s) {
    const uint8_t* sa = lds + (s % NA) * A_BYTES;
    const uint8_t* sbb = lds + (s % NB) * B_BYTES;
#pragma unroll
    for (int kk = 0; kk < 2; ++kk) {
      v4i t0 = *(const LDSP v4i*)(LDSP uint8_t*)(sa + x_addr[kk][0]);
      v4i t1 = *(const LDSP v4i*)(LDSP uint8_t*)(sa + x_addr[kk][1]);
      f.x[kk] = __builtin_shufflevector(t0, t1, 0, 1, 2, 3, 4, 5, 6, 7);
#pragma unroll
      for (int t = 0; t < 2; ++t)
#pragma unroll
        for (int r = 0; r < 4; ++r) {
          v2i a = __builtin_amdgcn_ds_read_tr8_b64_v2i32((LDSP v2i*)(LDSP uint8_t*)(sbb + w_addr[t] + kk * 8192 + r * 1024));
          f.w[t][kk][2 * r] = a[0]; f.w[t][kk][2 * r + 1] = a[1];
        }
    }
  };
  int avail = 0;
  auto wait_full = [&](int s) {
    while (avail <= s) {
      avail = lds_min8(lds_base + LDS_SYNC);
    }
  };
  v16f acc0 = {}, acc1 = {};
  auto mma = [&](const Frag& f) {
#pragma unroll
    for (int kk = 0; kk < 2; ++kk) {
      acc0 = __builtin_amdgcn_mfma_scale_f32_32x32x64_f8f6f4(f.w[0][kk], f.x[kk], acc0, 1, 0, 0, 0x7F7F7F7F, 0, 0x7F7F7F7F);
      acc1 = __builtin_amdgcn_mfma_scale_f32_32x32x64_f8f6f4(f.w[1][kk], f.x[kk], acc1, 1, 0, 0, 0x7F7F7F7F, 0, 0x7F7F7F7F);
    }
  };

#define INTERLEAVE() do { \
    _Pragma("unroll") for (int _i = 0; _i < 4; ++_i) { \
      __builtin_amdgcn_sched_group_barrier(0x008, 1, 0); \
      __builtin_amdgcn_sched_group_barrier(0x100, 5, 0); } \
    __builtin_amdgcn_sched_barrier(0); \
    asm volatile("" : "+v"(acc0), "+v"(acc1)); } while (0)
  Frag f0, f1;
  wait_full(0);
  STAMP(2);
  load_frag(f0, 0);
#pragma unroll 1
  for (int s = 0; s < NSTEP - 2; s += 2) {
    if (s == 4) STAMP(3);
    wait_full(s + 1);
    load_frag(f1, s + 1);
    mma(f0);
    INTERLEAVE();
    WAIT_L0();
    if (lane == 0) sync[8 + wave] = s + 2;
    wait_full(s + 2);
    load_frag(f0, s + 2);
    mma(f1);
    INTERLEAVE();
    WAIT_L0();
    if (lane == 0) sync[8 + wave] = s + 3;
  }
  wait_full(NSTEP - 1);
  load_frag(f1, NSTEP - 1);
  mma(f0);
  mma(f1);
  STAMP(4);
  TRACE_DUMP();

  const bool anybad = (sync[12] != 0) || (__hip_atomic_load(tmo, __ATOMIC_RELAXED, __HIP_MEMORY_SCOPE_AGENT) == call);
  {
    const float sc = p.sx[0] * p.sw[0];
    const int m = rb * BM + wm * 32 + (lane & 31);
    const int n0 = nb * BN + wn * 64 + (lane >> 5) * 4;
    float* op = p.out + (size_t)m * NDIM + n0;
#pragma unroll
    for (int t = 0; t < 2; ++t)
#pragma unroll
      for (int j = 0; j < 4; ++j) {
        v4f o;
#pragma unroll
        for (int e = 0; e < 4; ++e) {
          float y = (t == 0 ? acc0[4 * j + e] : acc1[4 * j + e]) * sc;
          y = fmaxf(y, 0.0f);
          o[e] = anybad ? __builtin_nanf("") : y;
        }
        *(v4f*)(op + t * 32 + j * 8) = o;
      }
  }
  if (tid == 0) __hip_atomic_store(ctr, call, __ATOMIC_RELAXED, __HIP_MEMORY_SCOPE_AGENT);
  STAMP(5);
}

extern "C" void kernel_launch(int rank, int nranks, ncclComm_t comm,
                              hipStream_t stream,
                              void* const* d_in, const int* in_sizes, int n_in,
                              void* d_out, int out_size,
                              const int* in_dtypes, int out_dtype,
                              void* const* d_out_all,
                              void* const* d_ws_all, size_t ws_bytes) {
  (void)comm; (void)in_sizes; (void)n_in; (void)out_size; (void)in_dtypes; (void)out_dtype; (void)d_out_all;
  if (nranks != NR || ws_bytes < WS_NEED) { fprintf(stderr, "unexpected config\n"); return; }
  static bool attr_done = false;
  if (!attr_done) {
    HIP_CHECK(hipFuncSetAttribute((const void*)a2a_gemm_kernel, hipFuncAttributeMaxDynamicSharedMemorySize, LDS_BYTES));
    attr_done = true;
  }
  Params p;
  p.x = (const uint8_t*)d_in[0];
  p.w = (const uint8_t*)d_in[1];
  p.sx = (const float*)d_in[2];
  p.sw = (const float*)d_in[3];
  p.out = (float*)d_out;
  for (int i = 0; i < NR; ++i) p.ws[i] = (uint8_t*)d_ws_all[i];
  p.rank = rank;
  a2a_gemm_kernel<<<NWG, NTHREADS, LDS_BYTES, stream>>>(p);
  HIP_CHECK(hipGetLastError());
}
```

```cpp
#include <hip/hip_runtime.h>
#include <rccl/rccl.h>
#include <cstdio>
#include <cstdint>
#include <type_traits>
#define ABL_PUSHFIRST

#define HIP_CHECK(expr) do { hipError_t _e = (expr); if (_e != hipSuccess) { \
    fprintf(stderr, "HIP error %s at %s:%d\n", hipGetErrorString(_e), __FILE__, __LINE__); \
    return; } } while (0)

typedef int v2i __attribute__((ext_vector_type(2)));
typedef int v4i __attribute__((ext_vector_type(4)));
typedef int v8i __attribute__((ext_vector_type(8)));
typedef float v4f __attribute__((ext_vector_type(4)));
typedef float v16f __attribute__((ext_vector_type(16)));
#define LDSP __attribute__((address_space(3)))
#define CSTP __attribute__((address_space(4)))

constexpr int NR = 8;
constexpr int MLOC = 512;
constexpr int KB = 512;
constexpr int NDIM = 4096;
constexpr int BM = 64, BN = 128;
constexpr int NWAVES = 7;
constexpr int NTHREADS = NWAVES * 64;
constexpr int NWG = 256;
constexpr int NST = 8;
constexpr int WV_BYTES = 40960;
constexpr int X_OFF = 0, W_OFF = 16384;
constexpr int LDS_BYTES = 4 * WV_BYTES;
constexpr int RT_STRIDE = 144, RT_BYTES = 32 * RT_STRIDE;

constexpr size_t WS_CTR = 0;
constexpr size_t WS_TMO = 64 * 1024;
constexpr size_t WS_FLAG = 128 * 1024;
constexpr size_t FLAG_RB = 4096 + 256;
constexpr size_t WS_STAGE = 1024 * 1024;
constexpr size_t BLK_BYTES = (size_t)MLOC * KB;
constexpr size_t WS_NEED = WS_STAGE + 2 * NR * BLK_BYTES;
constexpr unsigned long long DEADLINE_TICKS = 50000000ull;
#define PF_DELAY 48
#define POLL_SLEEP 8

struct Params {
  const uint8_t* x;
  const uint8_t* w;
  const float* sx;
  const float* sw;
  float* out;
  uint8_t* ws[NR];
  int rank;
};

#define WAIT_V(n) asm volatile("s_waitcnt vmcnt(" #n ")" ::: "memory")
#define WAIT_L0() asm volatile("s_waitcnt lgkmcnt(0)" ::: "memory")
#define FENCE() __builtin_amdgcn_sched_barrier(0)
#define START_BAR() do { asm volatile("" ::: "memory"); __builtin_amdgcn_s_barrier(); asm volatile("" ::: "memory"); } while (0)

#define SC1 " sc1"
template <int OFF, bool COH>
__device__ __forceinline__ void glds2(const void* sbase, unsigned v0, unsigned v1, unsigned lds_dst) {
  if (COH)
    asm volatile("s_mov_b32 m0, %3\n\ts_nop 0\n\t"
                 "global_load_lds_dwordx4 %0, %2 offset:%4" SC1 "\n\tglobal_load_lds_dwordx4 %1, %2 offset:%5" SC1
                 :: "v"(v0), "v"(v1), "s"(sbase), "s"(lds_dst), "n"(OFF), "n"(OFF + 1024) : "memory");
  else
    asm volatile("s_mov_b32 m0, %3\n\ts_nop 0\n\t"
                 "global_load_lds_dwordx4 %0, %2 offset:%4\n\tglobal_load_lds_dwordx4 %1, %2 offset:%5"
                 :: "v"(v0), "v"(v1), "s"(sbase), "s"(lds_dst), "n"(OFF), "n"(OFF + 1024) : "memory");
}

template <int OFF>
__device__ __forceinline__ void glds2c(int cond, const void* sbase, unsigned v0, unsigned v1, unsigned lds_dst) {
  asm volatile("s_cmp_eq_u32 %6, 0\n\ts_cbranch_scc1 1f\n\ts_mov_b32 m0, %3\n\ts_nop 0\n\t"
               "global_load_lds_dwordx4 %0, %2 offset:%4" SC1 "\n\tglobal_load_lds_dwordx4 %1, %2 offset:%5" SC1 "\n1:"
               :: "v"(v0), "v"(v1), "s"(sbase), "s"(lds_dst), "n"(OFF), "n"(OFF + 1024), "s"(cond) : "memory", "scc");
}

#define STAMP(i)

__global__ void __launch_bounds__(NTHREADS, 1) a2a_gemm_kernel(Params p) {
  extern __shared__ __attribute__((aligned(16))) uint8_t lds[];
  STAMP(0);
  const int tid = threadIdx.x;
  const int lane = tid & 63;
  const int wave = __builtin_amdgcn_readfirstlane(tid >> 6);
  const int bid = blockIdx.x;
  const int rank = p.rank;
  uint8_t* const myws = p.ws[rank];

  const int g = bid & 7, idx = bid >> 3;
#define NBX 4
  constexpr int NBG = 32 / NBX, RBX = 32 / NBX;
  const int nb = (g % NBG) * NBX + idx % NBX;
  const int rb = (g / NBG) * RBX + idx / NBX;
  const int prow = lane >> 3;

  unsigned* ctr = (unsigned*)(myws + WS_CTR + (size_t)bid * 128);
  unsigned* tmo = (unsigned*)(myws + WS_TMO);

  if (wave == 6) {
#define PF_MODE 0
    if (PF_MODE == 0) START_BAR();
    if (PF_MODE >= 2) __builtin_amdgcn_s_sleep(PF_DELAY);
    const int t = idx - 4;
    const int src = idx < 4 ? rank : ((rank + 1 + t % 7) & 7), c = idx < 4 ? idx : t / 7;
    const uint8_t* wb = p.w + (size_t)(src * KB + c * 128) * NDIM + g * 512;
    unsigned d[8];
#pragma unroll
    for (int j = 0; j < 8; ++j) {
      const int e = j * 64 + lane;
      d[j] = *(const unsigned*)(wb + (size_t)(e >> 2) * NDIM + (e & 3) * 128);
      if (PF_MODE == 3) asm volatile("" :: "v"(d[j]) : "memory");
    }
#pragma unroll
    for (int j = 0; j < 8; ++j) asm volatile("" :: "v"(d[j]));
    STAMP(1);
    return;
  }

  const unsigned call = *(const CSTP unsigned*)(uintptr_t)ctr + 1u;
  const unsigned par = call & 1u;

  if (wave >= 4) {
    const int job = bid + NWG * (wave - 4);
    const unsigned dst = (unsigned)(job >> 6);
    const int rg = job & 63;
    if (dst < (unsigned)NR && (int)dst != rank) {
      const int row = rg * 8 + prow;
      const uint8_t* src = p.x + ((size_t)dst * MLOC + row) * KB + (lane & 7) * 16;
      v4i v[4];
#pragma unroll
      for (int c = 0; c < 4; ++c) v[c] = *(const v4i*)(src + c * 128);
      uint8_t* dptr = p.ws[dst] + WS_STAGE + ((size_t)par * NR + rank) * BLK_BYTES + (size_t)row * KB + (lane & 7) * 16;
      unsigned* flag = (unsigned*)(p.ws[dst] + WS_FLAG + (size_t)(rg >> 3) * FLAG_RB) + rank * 8 + (rg & 7);
      const unsigned fbase = (call - 1u) * 4u;
      asm volatile("" :: "v"(v[0]), "v"(v[1]), "v"(v[2]), "v"(v[3]) : "memory");
#define PUSH_ST(c) asm volatile("global_store_dwordx4 %0, %1, off sc0 sc1\n\ts_nop 1" :: "v"(dptr + (c) * 128), "v"(v[c]) : "memory")
#define PUSH_FL(n) do { if (lane == 0) __hip_atomic_store(flag, fbase + (n), __ATOMIC_RELAXED, __HIP_MEMORY_SCOPE_SYSTEM); } while (0)
      STAMP(2);
      PUSH_ST(0);
      WAIT_V(0); PUSH_FL(1u); STAMP(3); PUSH_ST(1);
      WAIT_V(0); PUSH_FL(2u); STAMP(4); PUSH_ST(2);
      WAIT_V(0); PUSH_FL(3u); STAMP(5); PUSH_ST(3);
      WAIT_V(0); PUSH_FL(4u);
      START_BAR();
    } else {
      START_BAR();
    }
    STAMP(1);
    return;
  }

  const unsigned lds0 = (unsigned)(uintptr_t)(LDSP uint8_t*)lds;
  const unsigned lbase = lds0 + (unsigned)wave * WV_BYTES;
  const uint8_t* const a_local = p.x + (size_t)rank * MLOC * KB;
  const uint8_t* const a_stage = myws + WS_STAGE + (size_t)par * NR * BLK_BYTES;
  const unsigned* const myflag = (const unsigned*)(myws + WS_FLAG + (size_t)rb * FLAG_RB) + lane;
  const unsigned need_base = (call - 1u) * 4u;

  unsigned w_voff[8], x_voff[8];
#pragma unroll
  for (int q = 0; q < 8; ++q) {
    w_voff[q] = (unsigned)((q * 8 + prow) * NDIM + nb * BN + (((lane & 7) ^ (((prow >> 1) & 3) << 1)) * 16) - (q & 3) * 1024);
    x_voff[q] = (unsigned)((rb * BM + q * 8 + prow) * KB + (((lane & 7) ^ ((q & 1) * 4 + (prow >> 1))) * 16) - (q & 3) * 1024);
  }

  auto rstep = [&](int s, int& src, int& c) {
    const int t = (s - 1) * 4 + wave; c = (t * 37) >> 8; src = (rank + 1 + t - c * 7) & 7;
  };
  auto w_ptr = [&](int j) -> const uint8_t* {
    int src, c; rstep(j >> 1, src, c);
    return p.w + (size_t)(unsigned)(src * KB + c * 128 + (j & 1) * 64) * NDIM;
  };
  auto w2 = [&](const uint8_t* wp, unsigned dst, auto h_) {
    constexpr int h = decltype(h_)::value;
    glds2<(h & 1) * 2048, false>(wp, w_voff[2 * h], w_voff[2 * h + 1], dst + (h >> 1) * 4096);
  };
  auto x2 = [&](const uint8_t* xp, unsigned dst, auto h_) {
    constexpr int h = decltype(h_)::value;
    glds2<(h & 1) * 2048, true>(xp, x_voff[2 * h], x_voff[2 * h + 1], dst + (h >> 1) * 4096);
  };
  auto x2c = [&](int cond, const uint8_t* xp, unsigned dst, auto h_) {
    constexpr int h = decltype(h_)::value;
    glds2c<(h & 1) * 2048>(cond, xp, x_voff[2 * h], x_voff[2 * h + 1], dst + (h >> 1) * 4096);
  };
  using I0 = std::integral_constant<int, 0>; using I1 = std::integral_constant<int, 1>;
  using I2 = std::integral_constant<int, 2>; using I3 = std::integral_constant<int, 3>;
  using I8 = std::integral_constant<int, 8>; using IM = std::integral_constant<int, -1>;
  auto issue_w = [&](const uint8_t* wp, unsigned dst) { w2(wp, dst, I0{}); w2(wp, dst, I1{}); w2(wp, dst, I2{}); w2(wp, dst, I3{}); };
  auto issue_x = [&](const uint8_t* xp, unsigned dst) { x2(xp, dst, I0{}); x2(xp, dst, I1{}); x2(xp, dst, I2{}); x2(xp, dst, I3{}); };
  unsigned fc[2] = {0u, 0u};
  bool bad = false;
#define FLAG_LOAD(r) asm volatile("global_load_dword %0, %1, off sc0 sc1" : "+v"(r) : "v"(myflag) : "memory")
#define WNF 25
#define FLAG_TIE(r) asm volatile("" : "+v"(r))

  START_BAR();
  {
    const uint8_t* w0 = p.w + (size_t)(unsigned)(rank * KB + wave * 128) * NDIM;
    issue_w(w0, lbase + W_OFF);
    issue_x(a_local + wave * 128, lbase + X_OFF);
    issue_w(w0 + (size_t)64 * NDIM, lbase + W_OFF + 8192);
    FLAG_LOAD(fc[0]);
    issue_w(w_ptr(2), lbase + W_OFF + 16384);
    FLAG_LOAD(fc[1]);
  }

  const int xrow = lane & 31;
  const int xsw = (xrow >> 1) & 7;
  unsigned xa[2][2];
#pragma unroll
  for (int kk = 0; kk < 2; ++kk)
#pragma unroll
    for (int h = 0; h < 2; ++h) xa[kk][h] = lbase + X_OFF + xrow * 128 + (((kk * 4 + (lane >> 5) * 2 + h) ^ xsw) * 16);
  const int q8 = (lane & 15) >> 1, pp = lane & 1;
  const int wsw = ((q8 >> 1) & 3) << 1;
  unsigned wa[4];
#pragma unroll
  for (int t = 0; t < 4; ++t)
    wa[t] = lbase + W_OFF + ((lane >> 5) * 32 + q8) * 128 + (((2 * t + ((lane >> 4) & 1)) ^ wsw) * 16) + pp * 8;

  v8i wf[4] = {}, xf[3][2] = {};
  auto read_w = [&](int t) {
#pragma unroll
    for (int r = 0; r < 4; ++r) {
      v2i a = __builtin_amdgcn_ds_read_tr8_b64_v2i32((LDSP v2i*)(uintptr_t)(wa[t] + r * 1024));
      wf[t][2 * r] = a[0]; wf[t][2 * r + 1] = a[1];
    }
  };
  auto read_x = [&](int kk, int b) {
#pragma unroll
    for (int mt = 0; mt < 2; ++mt) {
      v4i t0 = *(const LDSP v4i*)(uintptr_t)(xa[kk][0] + mt * 4096);
      v4i t1 = *(const LDSP v4i*)(uintptr_t)(xa[kk][1] + mt * 4096);
      xf[b][mt] = __builtin_shufflevector(t0, t1, 0, 1, 2, 3, 4, 5, 6, 7);
    }
  };
  v16f acc[2][4];
#pragma unroll
  for (int mt = 0; mt < 2; ++mt)
#pragma unroll
    for (int t = 0; t < 4; ++t) acc[mt][t] = v16f{};
  auto mf = [&](int t, int mt, int kk) {
    FENCE();
    acc[mt][t] = __builtin_amdgcn_mfma_scale_f32_32x32x64_f8f6f4(wf[t], xf[kk][mt], acc[mt][t], 1, 0, 0, 0x7F7F7F7F, 0, 0x7F7F7F7F);
    asm volatile("" : "+v"(acc[mt][t]));
    FENCE();
  };

  int xn = 1, xn1 = 1, xn2 = 1;
  unsigned wslot = 0;
  int xdelta = 8192;
  int gx = 0;
  bool allready = false;
  const unsigned long long own = 0xFFull << (rank * 8);

  auto sub = [&](int s, auto H_, auto MODE_, auto WN_, auto RD_) {
    constexpr int H = decltype(H_)::value;
    constexpr int MODE = decltype(MODE_)::value;
    constexpr int WN = decltype(WN_)::value;
    constexpr bool RD = decltype(RD_)::value != 0;
    const int j = 2 * s + H;
    mf(0, 0, H); mf(0, 1, H);
    if (RD) {
      WAIT_L0();
      if (WN < 0) asm volatile("s_cmp_lg_u32 %0, 0\n\ts_cbranch_scc1 1f\n\ts_waitcnt vmcnt(8)\n1:\n\ts_waitcnt vmcnt(16)" :: "s"(gx) : "memory", "scc");
      else asm volatile("s_waitcnt vmcnt(%0)" :: "n"(WN < 0 ? 0 : WN) : "memory");
      FLAG_TIE(fc[H]);
      if (H == 1) {
        if (__builtin_expect(xn2 <= s + 1, 0)) {
          if (xn <= s + 1) {
            int src, c; rstep(s + 1, src, c);
            const unsigned long long want = 0xFFull << (src * 8);
            const unsigned need = need_base + (unsigned)c + 1u;
            WAIT_V(0);
            if (!bad && !allready && (__ballot(fc[1] >= need) & want) != want) {
              const unsigned long long t0 = __builtin_amdgcn_s_memrealtime();
              for (;;) {
                asm volatile("s_mov_b64 exec, %2\n\tglobal_load_dword %0, %1, off sc0 sc1\n\ts_mov_b64 exec, -1\n\ts_waitcnt vmcnt(0)"
                             : "+v"(fc[1]) : "v"(myflag), "s"(want) : "memory");
                if ((__ballot(fc[1] >= need) & want) == want) break;
                if (__builtin_amdgcn_s_memrealtime() - t0 > DEADLINE_TICKS) {
                  if (lane == 0) __hip_atomic_store(tmo, call, __ATOMIC_RELAXED, __HIP_MEMORY_SCOPE_AGENT);
                  bad = true;
                  break;
                }
                __builtin_amdgcn_s_sleep(POLL_SLEEP);
              }
            }
            issue_x(a_stage + (size_t)(unsigned)(src * (int)BLK_BYTES + c * 128), lbase + X_OFF + (unsigned)((s + 1) & 1) * 8192);
            xn = s + 2;
          }
          WAIT_V(0);
        }
      }
    }
    const unsigned wdst = lbase + W_OFF + wslot;
    const unsigned slot_nxt = wslot == 16384 ? 0u : wslot + 8192u;
    const int wd = (int)slot_nxt - (int)wslot;
    wslot = slot_nxt;
    if (RD) {
#pragma unroll
      for (int t = 0; t < 4; ++t) wa[t] += wd;
      read_w(0);
      if (H == 1) {
#pragma unroll
        for (int kk = 0; kk < 2; ++kk) { xa[kk][0] += xdelta; xa[kk][1] += xdelta; }
        xdelta = -xdelta;
        read_x(0, 0); read_x(1, 2);
      }
    }
    const uint8_t* wp = MODE >= 1 ? w_ptr(j + 3) : nullptr;
    int ok = 0, more_flags = 0;
    const uint8_t* xp = nullptr;
    unsigned xdst = 0;
    if (MODE == 2) {
      int src, c; rstep(xn < NST ? xn : 1, src, c);
      const unsigned long long want = 0xFFull << (src * 8);
      const bool match = (__ballot(fc[H] >= need_base + (unsigned)c + 1u) & want) == want;
      ok = __builtin_amdgcn_readfirstlane(((xn <= s + 2) & (xn < NST) & (allready | bad | match)) ? 1 : 0);
      allready = allready | bad | ((__ballot(fc[H] >= need_base + 4u) | own) == ~0ull);
      more_flags = __builtin_amdgcn_readfirstlane(allready ? 0 : 1);
      xp = a_stage + (size_t)(unsigned)(src * (int)BLK_BYTES + c * 128);
      xdst = lbase + X_OFF + (unsigned)(xn & 1) * 8192;
    }
    if (MODE >= 1) w2(wp, wdst, I0{});
    mf(1, 0, H);
    if (MODE >= 1) w2(wp, wdst, I1{});
    mf(1, 1, H);
    if (RD) read_w(1);
    if (MODE >= 1) w2(wp, wdst, I2{});
    mf(2, 0, H);
    if (MODE >= 1) w2(wp, wdst, I3{});
    mf(2, 1, H);
    if (RD) read_w(2);
    if (MODE == 2) { x2c(ok, xp, xdst, I0{}); x2c(ok, xp, xdst, I1{}); }
    mf(3, 0, H);
    if (MODE == 2) x2c(ok, xp, xdst, I2{});
    mf(3, 1, H);
    if (RD) read_w(3);
    if (MODE == 2) {
      x2c(ok, xp, xdst, I3{});
      asm volatile("s_cmp_eq_u32 %2, 0\n\ts_cbranch_scc1 1f\n\tglobal_load_dword %0, %1, off sc0 sc1\n1:"
                   : "+v"(fc[H]) : "v"(myflag), "s"(more_flags) : "memory", "scc");
    }
    xn2 = xn1; xn += ok; xn1 = xn; gx = ok;
    if (RD && H == 1) { xf[1][0] = xf[2][0]; xf[1][1] = xf[2][1]; }
  };

  WAIT_V(16);
  STAMP(2);
  read_w(0); read_x(0, 0); read_x(1, 1);
  FENCE();
  read_w(1); read_w(2); read_w(3);
#pragma unroll 1
  for (int s = 0; s < NST - 2; ++s) {
    sub(s, I0{}, I2{}, IM{}, I1{});
    sub(s, I1{}, I2{}, IM{}, I1{});
  }
  sub(NST - 2, I0{}, I1{}, IM{}, I1{});
  sub(NST - 2, I1{}, I0{}, I8{}, I1{});
  sub(NST - 1, I0{}, I0{}, I0{}, I1{});
  sub(NST - 1, I1{}, I0{}, I0{}, I0{});
  STAMP(4);

  {
    const unsigned wb = lbase + (lane & 31) * RT_STRIDE + (lane >> 5) * 16;
#pragma unroll
    for (int t = 0; t < 4; ++t)
#pragma unroll
      for (int mt = 0; mt < 2; ++mt)
#pragma unroll
        for (int jj = 0; jj < 4; ++jj) {
          v4f o = {acc[mt][t][4 * jj], acc[mt][t][4 * jj + 1], acc[mt][t][4 * jj + 2], acc[mt][t][4 * jj + 3]};
          *(LDSP v4f*)(uintptr_t)(wb + (t * 2 + mt) * RT_BYTES + jj * 32) = o;
        }
    *(LDSP int*)(uintptr_t)(lbase + 8 * RT_BYTES) = bad ? 1 : 0;
  }
  const float sc = p.sx[0] * p.sw[0];
  WAIT_L0();
  __builtin_amdgcn_s_barrier();
  {
    int anyb = 0;
#pragma unroll
    for (int w2 = 0; w2 < 4; ++w2) anyb |= *(const LDSP int*)(uintptr_t)(lds0 + w2 * WV_BYTES + 8 * RT_BYTES);
    const float scl = anyb ? __builtin_nanf("") : sc;
    const unsigned rbase = lds0 + wave * 2 * RT_BYTES + prow * RT_STRIDE + (lane & 7) * 16;
    float* op = p.out + (size_t)(rb * BM + prow) * NDIM + nb * BN + wave * 32 + (lane & 7) * 4;
#pragma unroll
    for (int mt = 0; mt < 2; ++mt)
#pragma unroll
      for (int i = 0; i < 4; ++i) {
        v4f sum = {};
#pragma unroll
        for (int w2 = 0; w2 < 4; ++w2)
          sum += *(const LDSP v4f*)(uintptr_t)(rbase + w2 * WV_BYTES + mt * RT_BYTES + i * 8 * RT_STRIDE);
        v4f o;
#pragma unroll
        for (int e = 0; e < 4; ++e) { const float y = sum[e] * scl; o[e] = anyb ? y : fmaxf(y, 0.0f); }
        *(v4f*)(op + (size_t)(mt * 32 + i * 8) * NDIM) = o;
      }
  }
  if (tid == 0) __hip_atomic_store(ctr, call, __ATOMIC_RELAXED, __HIP_MEMORY_SCOPE_AGENT);
  STAMP(5);
}

extern "C" void kernel_launch(int rank, int nranks, ncclComm_t comm,
                              hipStream_t stream,
                              void* const* d_in, const int* in_sizes, int n_in,
                              void* d_out, int out_size,
                              const int* in_dtypes, int out_dtype,
                              void* const* d_out_all,
                              void* const* d_ws_all, size_t ws_bytes) {
  (void)comm; (void)in_sizes; (void)n_in; (void)out_size; (void)in_dtypes; (void)out_dtype; (void)d_out_all;
  if (nranks != NR || ws_bytes < WS_NEED) { fprintf(stderr, "unexpected config\n"); return; }
  static bool attr_done = false;
  if (!attr_done) {
    HIP_CHECK(hipFuncSetAttribute((const void*)a2a_gemm_kernel, hipFuncAttributeMaxDynamicSharedMemorySize, LDS_BYTES));
    attr_done = true;
  }
  Params p;
  p.x = (const uint8_t*)d_in[0];
  p.w = (const uint8_t*)d_in[1];
  p.sx = (const float*)d_in[2];
  p.sw = (const float*)d_in[3];
  p.out = (float*)d_out;
  for (int i = 0; i < NR; ++i) p.ws[i] = (uint8_t*)d_ws_all[i];
  p.rank = rank;
  a2a_gemm_kernel<<<NWG, NTHREADS, LDS_BYTES, stream>>>(p);
  HIP_CHECK(hipGetLastError());
}
```

```cpp
#include <hip/hip_runtime.h>
#include <rccl/rccl.h>
#include <cstdio>
#include <cstdint>
#include <type_traits>
#define SYNC_MODE 2

#define HIP_CHECK(expr) do { hipError_t _e = (expr); if (_e != hipSuccess) { \
    fprintf(stderr, "HIP error %s at %s:%d\n", hipGetErrorString(_e), __FILE__, __LINE__); \
    return; } } while (0)

typedef int v2i __attribute__((ext_vector_type(2)));
typedef int v4i __attribute__((ext_vector_type(4)));
typedef int v8i __attribute__((ext_vector_type(8)));
typedef float v4f __attribute__((ext_vector_type(4)));
typedef float v16f __attribute__((ext_vector_type(16)));
#define LDSP __attribute__((address_space(3)))
#define CSTP __attribute__((address_space(4)))

constexpr int NR = 8;
constexpr int MLOC = 512;
constexpr int KB = 512;
constexpr int NDIM = 4096;
constexpr int BM = 64, BN = 128;
#define PF_SER 1
constexpr int NWAVES = PF_SER > 0 ? 6 : 5;
constexpr int NTHREADS = NWAVES * 64;
constexpr int NWG = 256;
constexpr int NST = 8;
constexpr int WV_BYTES = 40960;
constexpr int X_OFF = 0, W_OFF = 16384;
constexpr int LDS_BYTES = 4 * WV_BYTES;
constexpr int RT_STRIDE = 144, RT_BYTES = 32 * RT_STRIDE;

constexpr size_t WS_CTR = 0;
constexpr size_t WS_TMO = 64 * 1024;
constexpr size_t WS_FLAG = 128 * 1024;
constexpr size_t FLAG_STRIDE = 128;
constexpr size_t WS_STAGE = 1024 * 1024;
constexpr size_t BLK_BYTES = (size_t)MLOC * KB;
constexpr size_t WS_NEED = WS_STAGE + 2 * NR * BLK_BYTES;
constexpr unsigned long long DEADLINE_TICKS = 50000000ull;
#define GEMM_DELAY 0
#define PF_DELAY 0
#define STAG 0
#define PUSH_DT 128
#define PUSH_T0 170
#define PUSH_DR 1
#define PUSH_H0 0
#define PUSH_HB 0
#define PUSH_HS 2
#define PUSH_X 0
#define EARLY_POLL 0
#define OUT_MODE 3
#define ADAPT 120
#define ADAPT_C 1
#define ADAPT_SLEEP 10
#define OUT_WIDE 0
#define PUSH_NT ""
#define TLB_WARM 0
#define EPI2 3
#define OUT_WIN 0
#define OUT_SLEEP 0
#define PUSH_DS 16
#define HOLD_C 1
#define GATE 200
#define PUSH_WIN 2
#define POLL_SLEEP 2

struct Params {
  const uint8_t* x;
  const uint8_t* w;
  const float* sx;
  const float* sw;
  float* out;
  uint8_t* myws;
  int rank;
  int pad;
  uint8_t* ws[NR];
};

#define WAIT_V(n) asm volatile("s_waitcnt vmcnt(" #n ")" ::: "memory")
#define WAIT_L0() asm volatile("s_waitcnt lgkmcnt(0)" ::: "memory")
#define FENCE() __builtin_amdgcn_sched_barrier(0)
#define START_BAR() do { asm volatile("" ::: "memory"); __builtin_amdgcn_s_barrier(); asm volatile("" ::: "memory"); } while (0)

#define SC1 " sc1"
template <int OFF, bool COH>
__device__ __forceinline__ void glds2(const void* sbase, unsigned v0, unsigned v1, unsigned lds_dst) {
  if (COH)
    asm volatile("s_mov_b32 m0, %3\n\ts_nop 0\n\t"
                 "global_load_lds_dwordx4 %0, %2 offset:%4" SC1 "\n\tglobal_load_lds_dwordx4 %1, %2 offset:%5" SC1
                 :: "v"(v0), "v"(v1), "s"(sbase), "s"(lds_dst), "n"(OFF), "n"(OFF + 1024) : "memory");
  else
    asm volatile("s_mov_b32 m0, %3\n\ts_nop 0\n\t"
                 "global_load_lds_dwordx4 %0, %2 offset:%4\n\tglobal_load_lds_dwordx4 %1, %2 offset:%5"
                 :: "v"(v0), "v"(v1), "s"(sbase), "s"(lds_dst), "n"(OFF), "n"(OFF + 1024) : "memory");
}

template <int OFF>
__device__ __forceinline__ void glds2c(unsigned long long cond, const void* sbase, unsigned v0, unsigned v1, unsigned lds_dst) {
  asm volatile("s_cmp_eq_u64 %6, 0\n\ts_cbranch_scc1 1f\n\ts_mov_b32 m0, %3\n\ts_nop 0\n\t"
               "global_load_lds_dwordx4 %0, %2 offset:%4" SC1 "\n\tglobal_load_lds_dwordx4 %1, %2 offset:%5" SC1 "\n1:"
               :: "v"(v0), "v"(v1), "s"(sbase), "s"(lds_dst), "n"(OFF), "n"(OFF + 1024), "s"(cond) : "memory", "scc");
}

#define STAMP(i)
#define TS(i)
#define TS_FLUSH()

__global__ void __launch_bounds__(NTHREADS, 1) a2a_gemm_kernel(Params p) {
  extern __shared__ __attribute__((aligned(16))) uint8_t lds[];
  STAMP(0);
  const int tid = threadIdx.x;
  const int lane = tid & 63;
  const int wave = __builtin_amdgcn_readfirstlane(tid >> 6);
  const int bid = blockIdx.x;
  const unsigned long long t_start = __builtin_amdgcn_s_memrealtime();
  const int rank = p.rank;
  uint8_t* const myws = p.myws;

  const int g = bid & 7, idx = bid >> 3;
#define NBX 4
  constexpr int NBG = 32 / NBX, RBX = 32 / NBX;
  const int nb = (g % NBG) * NBX + idx % NBX;
  const int rb = (g / NBG) * RBX + idx / NBX;
  const int prow = lane >> 3;

  unsigned* ctr = (unsigned*)(myws + WS_CTR + (size_t)bid * 128);
  unsigned* tmo = (unsigned*)(myws + WS_TMO);

  if (wave == 5) {
    START_BAR();
    const int e = (idx & 7) * 64 + lane;
    const uint8_t* wb = p.w + (size_t)(e >> 2) * NDIM + g * 512 + (e & 3) * 128;
    unsigned d[7];
#pragma unroll
    for (int j = 0; j < 7; ++j) {
      const int t = 4 * j + (idx >> 3), c = (t * 37) >> 8, src = (rank + 1 + t - c * 7) & 7;
      d[j] = *(const unsigned*)(wb + (size_t)(unsigned)((src * KB + c * 128) * NDIM));
      if (j >= PF_SER - 1) asm volatile("" :: "v"(d[j - PF_SER + 1]) : "memory");
    }
#pragma unroll
    for (int j = 0; j < 7; ++j) asm volatile("" :: "v"(d[j]));
    STAMP(1);
    return;
  }

  asm volatile("" :: "s"(p.x), "s"(p.w), "s"(rank) : "memory");
  unsigned ctr_raw = *(const CSTP unsigned*)(uintptr_t)ctr;
#define GET_CALL() asm volatile("" : "+s"(ctr_raw) :: "memory"); \
  const unsigned call = ctr_raw + 1u; const unsigned par = call & 1u

  if (wave == 4) {
    const int jd = bid & 7, jr = (bid >> 3) & 7, jc = bid >> 6;
    const unsigned dst = (unsigned)(rank + 1 + ((jd + jr) & 7)) & 7u;
    if ((int)dst != rank) {
      const int row = jr * 64 + prow;
      const uint8_t* src = p.x + ((size_t)dst * MLOC + row) * KB + jc * 128 + (lane & 7) * 16;
      v4i v[8];
#pragma unroll
      for (int i = 0; i < 8; ++i) v[i] = *(const v4i*)(src + i * 8 * KB);
      const bool early = jc >= HOLD_C || (PUSH_H0 > 0 && PUSH_HB && (jd >> PUSH_HS) != 0);
      if (early) START_BAR();
      GET_CALL();
      uint8_t* dptr = p.ws[dst] + WS_STAGE + ((size_t)par * NR + rank) * BLK_BYTES + (size_t)row * KB + jc * 128 + (lane & 7) * 16;
      uint8_t* flag = p.ws[dst] + WS_FLAG + (size_t)((jr * 8 + rank) * 4 + jc) * FLAG_STRIDE;
      asm volatile("" :: "v"(v[0]), "v"(v[1]), "v"(v[2]), "v"(v[3]), "v"(v[4]), "v"(v[5]), "v"(v[6]), "v"(v[7]) : "memory");
      if (PUSH_X == 0 && jc > 0) {
        if (jc < HOLD_C) START_BAR();
        const unsigned long long gate = (unsigned long long)(PUSH_T0 + jc * PUSH_DT + (PUSH_DR ? jr : jd) * PUSH_DS);
        const uint8_t* mirror = myws + WS_FLAG + (size_t)((jr * 8 + dst) * 4 + jc - ADAPT_C) * FLAG_STRIDE;
        for (;;) {
          const unsigned long long el = __builtin_amdgcn_s_memrealtime() - t_start;
          if (el >= gate) break;
          if (ADAPT > 0 && el + ADAPT < gate) {
            unsigned f;
            asm volatile("global_load_dword %0, %1, off sc0 sc1\n\ts_waitcnt vmcnt(0)" : "=v"(f) : "v"(mirror) : "memory");
            if ((unsigned)__builtin_amdgcn_readfirstlane((int)f) >= call) break;
            __builtin_amdgcn_s_sleep(ADAPT_SLEEP);
          } else {
            __builtin_amdgcn_s_sleep(1);
          }
        }
      }
      STAMP(4);
#pragma unroll
      for (int i = 0; i < 8; ++i)
        asm volatile("global_store_dwordx4 %0, %1, off sc0 sc1" PUSH_NT "\n\ts_nop 1" :: "v"(dptr + i * 8 * KB), "v"(v[i]) : "memory");
      if (jc == 0 && !early) START_BAR();
      WAIT_V(0);
      asm volatile("s_mov_b64 exec, 1\n\tglobal_store_dword %0, %1, off sc0 sc1\n\ts_mov_b64 exec, -1" :: "v"(flag), "v"(call) : "memory");
    } else {
      START_BAR();
    }
    STAMP(1);
    return;
  }
  if (wave == 4) {
    const int jd = bid & 7, jr = (bid >> 3) & 7;
    const unsigned dst = (unsigned)(rank + 1 + ((jd + jr) & 7)) & 7u;
    if (bid < 64 && (int)dst != rank) {
      const int row = jr * 64 + prow;
      const uint8_t* src = p.x + ((size_t)dst * MLOC + row) * KB + (lane & 7) * 16;
      v4i v[32];
      TS(2);
#pragma unroll
      for (int i = 0; i < 32; ++i)
        asm volatile("global_load_dwordx4 %0, %1, off offset:%2" : "=v"(v[i]) : "v"(src + (i & 7) * 8 * KB), "n"((i >> 3) * 128) : "memory");
      GET_CALL();
      TS(3);
      uint8_t* dptr = p.ws[dst] + WS_STAGE + ((size_t)par * NR + rank) * BLK_BYTES + (size_t)row * KB + (lane & 7) * 16;
      uint8_t* flag = p.ws[dst] + WS_FLAG + (size_t)((jr * 8 + rank) * 4) * FLAG_STRIDE;
#define PUSH_ST(i) asm volatile("global_store_dwordx4 %0, %1, off offset:%2 sc0 sc1\n\ts_nop 1" :: "v"(dptr + ((i) & 7) * 8 * KB), "v"(v[i]), "n"(((i) >> 3) * 128) : "memory")
#define PUSH_H(h) do { PUSH_ST(4 * (h)); PUSH_ST(4 * (h) + 1); PUSH_ST(4 * (h) + 2); PUSH_ST(4 * (h) + 3); } while (0)
#define PUSH_FL(c) asm volatile("s_mov_b64 exec, 1\n\tglobal_store_dword %0, %1, off sc0 sc1\n\ts_mov_b64 exec, -1" :: "v"(flag + (c) * FLAG_STRIDE), "v"(call) : "memory")
      WAIT_V(28); PUSH_H(0); WAIT_V(28); PUSH_H(1);
      START_BAR();
      WAIT_V(4); PUSH_H(2);
      WAIT_V(4); PUSH_FL(0); PUSH_H(3);
      WAIT_V(5); PUSH_H(4);
      WAIT_V(4); PUSH_FL(1); PUSH_H(5);
      WAIT_V(5); PUSH_H(6);
      WAIT_V(4); PUSH_FL(2); PUSH_H(7);
      WAIT_V(0); PUSH_FL(3);
    } else {
      START_BAR();
    }
    STAMP(1);
    return;
  }

  const unsigned lds0 = (unsigned)(uintptr_t)(LDSP uint8_t*)lds;
  const unsigned lbase = lds0 + (unsigned)wave * WV_BYTES;
  const uint8_t* const a_local = p.x + (size_t)rank * MLOC * KB;
  auto rstep = [&](int k, int& src, int& c) {
    const int t = k * 4 + wave; c = (t * 37) >> 8; src = (rank + 1 + t - c * 7) & 7;
  };
  const unsigned* myflag;
  {
    int src, c; rstep(lane < 56 ? lane >> 3 : 6, src, c);
    myflag = (const unsigned*)(myws + WS_FLAG + (size_t)((rb * 8 + src) * 4 + c) * FLAG_STRIDE);
  }

  unsigned w_voff[8], x_voff[8];
#pragma unroll
  for (int q = 0; q < 8; ++q) {
    w_voff[q] = (unsigned)((q * 8 + prow) * NDIM + nb * BN + (((lane & 7) ^ (((prow >> 1) & 3) << 1)) * 16) - (q & 3) * 1024);
    x_voff[q] = (unsigned)((rb * BM + q * 8 + prow) * KB + (((lane & 7) ^ ((q & 1) * 4 + (prow >> 1))) * 16) - (q & 3) * 1024);
  }

  unsigned long long tab = 0;
#pragma unroll
  for (int k = 0; k < 7; ++k) { int src, c; rstep(k, src, c); tab |= (unsigned long long)(unsigned)(src * 4 + c) << (8 * k); }
  auto lookup = [&](unsigned long long bit) -> int { return (int)(tab >> __builtin_ctzll(bit | (1ull << 56))) & 31; };
  auto w_ptr = [&](int kq, int half) -> const uint8_t* {
    return p.w + (size_t)(unsigned)((kq * 128 + half * 64) * NDIM);
  };
  auto w2 = [&](const uint8_t* wp, unsigned dst, auto h_) {
    constexpr int h = decltype(h_)::value;
    glds2<(h & 1) * 2048, false>(wp, w_voff[2 * h], w_voff[2 * h + 1], dst + (h >> 1) * 4096);
  };
  auto x2 = [&](const uint8_t* xp, unsigned dst, auto h_) {
    constexpr int h = decltype(h_)::value;
    glds2<(h & 1) * 2048, true>(xp, x_voff[2 * h], x_voff[2 * h + 1], dst + (h >> 1) * 4096);
  };
  auto x2c = [&](unsigned long long cond, const uint8_t* xp, unsigned dst, auto h_) {
    constexpr int h = decltype(h_)::value;
    glds2c<(h & 1) * 2048>(cond, xp, x_voff[2 * h], x_voff[2 * h + 1], dst + (h >> 1) * 4096);
  };
  using I0 = std::integral_constant<int, 0>; using I1 = std::integral_constant<int, 1>;
  using I2 = std::integral_constant<int, 2>; using I3 = std::integral_constant<int, 3>;
  using I8 = std::integral_constant<int, 8>; using IM = std::integral_constant<int, -1>;
  auto issue_w = [&](const uint8_t* wp, unsigned dst) { w2(wp, dst, I0{}); w2(wp, dst, I1{}); w2(wp, dst, I2{}); w2(wp, dst, I3{}); };
  auto issue_x = [&](const uint8_t* xp, unsigned dst) { x2(xp, dst, I0{}); x2(xp, dst, I1{}); x2(xp, dst, I2{}); x2(xp, dst, I3{}); };
  unsigned fc[2] = {0u, 0u};
  bool bad = false;
#define FLAG_LOAD(r) asm volatile("global_load_dword %0, %1, off sc0 sc1" : "+v"(r) : "v"(myflag) : "memory")
#define WNF 25
#define FLAG_TIE(r) asm volatile("" : "+v"(r))

  STAMP(1);
  START_BAR();
  unsigned long long rem = 0x0001010101010101ull, pend, rdy = 0;
  int k1, k2 = 0;
  unsigned long long gx;
  {
    const uint8_t* w0 = p.w + (size_t)(unsigned)(rank * KB + wave * 128) * NDIM;
    if (SYNC_MODE < 2) FLAG_LOAD(fc[0]);
    issue_w(w0, lbase + W_OFF);
    issue_x(a_local + wave * 128, lbase + X_OFF);
    issue_w(w0 + (size_t)64 * NDIM, lbase + W_OFF + 8192);
  }
  GET_CALL();
  float sxv = *(const CSTP float*)(uintptr_t)p.sx, swv = *(const CSTP float*)(uintptr_t)p.sw;
  float* outp = p.out;
  asm volatile("" : "+s"(outp));
  const uint8_t* const a_stage = myws + WS_STAGE + (size_t)par * NR * BLK_BYTES;
  auto spread = [&](unsigned f) -> unsigned long long {
    return __ballot(f >= call) & 0x0101010101010101ull;
  };
  auto x_ptr = [&](int kq) -> const uint8_t* {
    return a_stage + (size_t)(unsigned)((kq >> 2) * (int)BLK_BYTES + (kq & 3) * 128);
  };
  {
    if (SYNC_MODE < 2) {
      WAIT_V(24);
      FLAG_TIE(fc[0]);
      rdy = spread(fc[0]) & rem;
    }
    if (SYNC_MODE == 3) {
      while (__builtin_amdgcn_s_memrealtime() - t_start < GATE) __builtin_amdgcn_s_sleep(2);
      for (;;) {
        asm volatile("global_load_dword %0, %1, off sc0 sc1\n\ts_waitcnt vmcnt(0)" : "+v"(fc[0]) : "v"(myflag) : "memory");
        rdy = spread(fc[0]) & rem;
        if (rdy != 0) break;
        if (__builtin_amdgcn_s_memrealtime() - t_start > DEADLINE_TICKS) {
          asm volatile("s_mov_b64 exec, 1\n\tglobal_store_dword %0, %1, off sc1\n\ts_mov_b64 exec, -1" :: "v"(tmo), "v"(call) : "memory");
          bad = true;
          rdy = rem;
          break;
        }
        __builtin_amdgcn_s_sleep(POLL_SLEEP);
      }
      STAMP(6);
    }
    gx = rdy;
    const unsigned long long b = rdy ? (rdy & -rdy) : 1ull;
    k1 = lookup(b);
    rem = rdy ? rem ^ b : rem;
    pend = rdy ? ~0ull : b;
    issue_w(w_ptr(k1, 0), lbase + W_OFF + 16384);
    if (SYNC_MODE != 2) {
      const uint8_t* xp = x_ptr(k1);
      x2c(gx, xp, lbase + X_OFF + 8192, I0{}); x2c(gx, xp, lbase + X_OFF + 8192, I1{});
      x2c(gx, xp, lbase + X_OFF + 8192, I2{}); x2c(gx, xp, lbase + X_OFF + 8192, I3{});
      FLAG_LOAD(fc[1]);
    }
  }

  const int xrow = lane & 31;
  const int xsw = (xrow >> 1) & 7;
  unsigned xa[2][2];
#pragma unroll
  for (int kk = 0; kk < 2; ++kk)
#pragma unroll
    for (int h = 0; h < 2; ++h) xa[kk][h] = lbase + X_OFF + xrow * 128 + (((kk * 4 + (lane >> 5) * 2 + h) ^ xsw) * 16);
  const int q8 = (lane & 15) >> 1, pp = lane & 1;
  const int wsw = ((q8 >> 1) & 3) << 1;
  unsigned wa[4];
#pragma unroll
  for (int t = 0; t < 4; ++t)
    wa[t] = lbase + W_OFF + ((lane >> 5) * 32 + q8) * 128 + (((2 * t + ((lane >> 4) & 1)) ^ wsw) * 16) + pp * 8;

  v8i wf[4] = {}, xf[3][2] = {};
  auto read_w = [&](int t) {
#pragma unroll
    for (int r = 0; r < 4; ++r) {
      v2i a = __builtin_amdgcn_ds_read_tr8_b64_v2i32((LDSP v2i*)(uintptr_t)(wa[t] + r * 1024));
      wf[t][2 * r] = a[0]; wf[t][2 * r + 1] = a[1];
    }
  };
  auto read_x = [&](int kk, int b) {
#pragma unroll
    for (int mt = 0; mt < 2; ++mt) {
      v4i t0 = *(const LDSP v4i*)(uintptr_t)(xa[kk][0] + mt * 4096);
      v4i t1 = *(const LDSP v4i*)(uintptr_t)(xa[kk][1] + mt * 4096);
      xf[b][mt] = __builtin_shufflevector(t0, t1, 0, 1, 2, 3, 4, 5, 6, 7);
    }
  };
  v16f acc[2][4];
#pragma unroll
  for (int mt = 0; mt < 2; ++mt)
#pragma unroll
    for (int t = 0; t < 4; ++t) acc[mt][t] = v16f{};
  auto mf = [&](int t, int mt, int kk) {
    FENCE();
    acc[mt][t] = __builtin_amdgcn_mfma_scale_f32_32x32x64_f8f6f4(wf[t], xf[kk][mt], acc[mt][t], 1, 0, 0, 0x7F7F7F7F, 0, 0x7F7F7F7F);
    asm volatile("" : "+v"(acc[mt][t]));
    FENCE();
  };

  int xn = gx != 0 ? 2 : 1, xn1 = xn, xn2 = 1;
  unsigned wslot = 0;
  int xdelta = 8192;

  auto sub = [&](int s, auto H_, auto MODE_, auto WN_, auto RD_) {
    constexpr int H = decltype(H_)::value;
    constexpr int MODE = decltype(MODE_)::value;
    constexpr int WN = decltype(WN_)::value;
    constexpr bool RD = decltype(RD_)::value != 0;
    mf(0, 0, H); mf(0, 1, H);
    if (RD) {
      WAIT_L0();
      if (WN < 0) asm volatile("s_cmp_lg_u64 %0, 0\n\ts_cbranch_scc1 1f\n\ts_waitcnt vmcnt(8)\n1:\n\ts_waitcnt vmcnt(16)" :: "s"(gx) : "memory", "scc");
      else asm volatile("s_waitcnt vmcnt(%0)" :: "n"(WN < 0 ? 0 : WN) : "memory");
      FLAG_TIE(fc[H]);
      if (H == 1) {
        if (__builtin_expect(xn2 <= s + 1, 0)) {
          if (xn <= s + 1) {
            WAIT_V(0);
            if (SYNC_MODE == 2 && s == 0) {
              while (__builtin_amdgcn_s_memrealtime() - t_start < GATE) __builtin_amdgcn_s_sleep(2);
            }
            FLAG_TIE(fc[0]);
            rdy |= spread(fc[0]);
            if ((rem & rdy) == 0) {
              const unsigned long long t0 = __builtin_amdgcn_s_memrealtime();
              for (;;) {
                asm volatile("global_load_dword %0, %1, off sc0 sc1\n\ts_waitcnt vmcnt(0)" : "+v"(fc[1]) : "v"(myflag) : "memory");
                rdy |= spread(fc[1]);
                if ((rem & rdy) != 0) break;
                if (__builtin_amdgcn_s_memrealtime() - t0 > DEADLINE_TICKS) {
                  asm volatile("s_mov_b64 exec, 1\n\tglobal_store_dword %0, %1, off sc1\n\ts_mov_b64 exec, -1" :: "v"(tmo), "v"(call) : "memory");
                  bad = true;
                  rdy = ~0ull;
                  break;
                }
                __builtin_amdgcn_s_sleep(POLL_SLEEP);
              }
            }
            if ((pend & rdy) == 0) {
              const unsigned long long cand = rem & rdy;
              pend = cand & -cand;
              k1 = lookup(pend);
              const unsigned s1 = wslot == 16384 ? 0u : wslot + 8192u, s2 = s1 == 16384 ? 0u : s1 + 8192u;
              issue_w(w_ptr(k1, 0), lbase + W_OFF + s1);
              issue_w(w_ptr(k1, 1), lbase + W_OFF + s2);
            }
            rem ^= pend;
            pend = ~0ull;
            issue_x(x_ptr(k1), lbase + X_OFF + (unsigned)((s + 1) & 1) * 8192);
            xn = s + 2;
          }
          WAIT_V(0);
        }
      }
    }
    const unsigned wdst = lbase + W_OFF + wslot;
    const unsigned slot_nxt = wslot == 16384 ? 0u : wslot + 8192u;
    const int wd = (int)slot_nxt - (int)wslot;
    wslot = slot_nxt;
    if (RD) {
#pragma unroll
      for (int t = 0; t < 4; ++t) wa[t] += wd;
      read_w(0);
      if (H == 1) {
#pragma unroll
        for (int kk = 0; kk < 2; ++kk) { xa[kk][0] += xdelta; xa[kk][1] += xdelta; }
        xdelta = -xdelta;
        read_x(0, 0); read_x(1, 2);
      }
    }
    int kx = 0;
    unsigned long long b = 0, ok = 0;
    if (H == 0) {
      const uint8_t* wp = MODE >= 1 ? w_ptr(k1, 1) : nullptr;
      if (MODE >= 1) w2(wp, wdst, I0{});
      mf(1, 0, H);
      if (MODE == 2) rdy |= spread(fc[H]);
      if (MODE >= 1) w2(wp, wdst, I1{});
      mf(1, 1, H);
      if (RD) read_w(1);
      if (MODE == 2) {
        const unsigned long long cand = rem & pend & rdy;
        b = cand & -cand;
        ok = b;
        kx = lookup(b);
      }
      if (MODE >= 1) w2(wp, wdst, I2{});
      mf(2, 0, H);
      if (MODE == 2) {
        rem ^= b;
        k2 = (b != 0 && xn == s + 2) ? kx : k2;
        pend = b != 0 ? ~0ull : pend;
      }
      if (MODE >= 1) w2(wp, wdst, I3{});
    } else {
      if (MODE == 2) rdy |= spread(fc[H]);
      mf(1, 0, H);
      const uint8_t* wp = nullptr;
      if (MODE == 2) {
        const bool allowed = xn == s + 2;
        const unsigned long long cand = rem & rdy, sel = cand ? cand : rem;
        b = sel & -sel;
        kx = lookup(b);
        ok = allowed ? (b & rdy) : 0ull;
        k2 = allowed ? kx : k2;
        wp = w_ptr(k2, 0);
        w2(wp, wdst, I0{});
      }
      mf(1, 1, H);
      if (RD) read_w(1);
      if (MODE == 2) { w2(wp, wdst, I1{}); w2(wp, wdst, I2{}); }
      mf(2, 0, H);
      if (MODE == 2) {
        pend = (xn == s + 2 && ok == 0) ? b : ~0ull;
        rem ^= ok;
        k1 = k2;
        w2(wp, wdst, I3{});
      }
    }
    const uint8_t* xp = x_ptr(kx);
    const unsigned xdst = lbase + X_OFF + (unsigned)(xn & 1) * 8192;
    mf(2, 1, H);
    if (RD) read_w(2);
    if (MODE == 2) { x2c(ok, xp, xdst, I0{}); x2c(ok, xp, xdst, I1{}); }
    mf(3, 0, H);
    const unsigned long long more_flags = (SYNC_MODE != 2 || (EARLY_POLL & 2) || H == 1 || s > 0) ? (rem & ~rdy) : 0ull;
    if (MODE == 2) x2c(ok, xp, xdst, I2{});
    mf(3, 1, H);
    if (RD) read_w(3);
    if (MODE == 2) {
      x2c(ok, xp, xdst, I3{});
      asm volatile("s_cmp_eq_u64 %2, 0\n\ts_cbranch_scc1 1f\n\tglobal_load_dword %0, %1, off sc0 sc1\n1:"
                   : "+v"(fc[H]) : "v"(myflag), "s"(more_flags) : "memory", "scc");
    }
    xn2 = xn1; xn += ok != 0 ? 1 : 0; xn1 = xn; gx = ok;
    if (RD && H == 1) { xf[1][0] = xf[2][0]; xf[1][1] = xf[2][1]; }
  };

  asm volatile("s_cmp_lg_u64 %0, 0\n\ts_cbranch_scc1 1f\n\ts_waitcnt vmcnt(16)\n1:\n\ts_waitcnt vmcnt(24)" :: "s"(gx) : "memory", "scc");
  STAMP(2);
  asm volatile("" : "+s"(sxv), "+s"(swv));
  read_w(0); read_x(0, 0); read_x(1, 1);
  FENCE();
  read_w(1); read_w(2); read_w(3);
#pragma unroll 1
  for (int s = 0; s < NST - 2; ++s) {
    sub(s, I0{}, I2{}, IM{}, I1{});
    sub(s, I1{}, I2{}, IM{}, I1{});
  }
  sub(NST - 2, I0{}, I1{}, IM{}, I1{});
  sub(NST - 2, I1{}, I0{}, I8{}, I1{});
  sub(NST - 1, I0{}, I0{}, I0{}, I1{});
  const unsigned wb = lbase + (lane & 31) * RT_STRIDE + (lane >> 5) * 16;
  auto wr = [&](int t) {
#pragma unroll
    for (int mt = 0; mt < 2; ++mt)
#pragma unroll
      for (int jj = 0; jj < 4; ++jj) {
        v4f o = {acc[mt][t][4 * jj], acc[mt][t][4 * jj + 1], acc[mt][t][4 * jj + 2], acc[mt][t][4 * jj + 3]};
        *(LDSP v4f*)(uintptr_t)(wb + (t * 2 + mt) * RT_BYTES + jj * 32) = o;
      }
  };
  mf(0, 0, 1); mf(0, 1, 1); mf(1, 0, 1);
  WAIT_L0();
  mf(1, 1, 1); wr(0);
  mf(2, 0, 1); mf(2, 1, 1); wr(1);
  mf(3, 0, 1); mf(3, 1, 1); wr(2); wr(3);
  STAMP(4);

  {
    *(LDSP int*)(uintptr_t)(lbase + 8 * RT_BYTES) = bad ? 1 : 0;
  }
  const float sc = sxv * swv;
  WAIT_L0();
  __builtin_amdgcn_s_barrier();
  {
    int braw[4];
#pragma unroll
    for (int w2 = 0; w2 < 4; ++w2) braw[w2] = *(const LDSP int*)(uintptr_t)(lds0 + w2 * WV_BYTES + 8 * RT_BYTES);
    const unsigned rbase = lds0 + wave * 2 * RT_BYTES + prow * RT_STRIDE + (lane & 7) * 16;
    float* op = outp + (size_t)(rb * BM + prow) * NDIM + nb * BN + wave * 32 + (lane & 7) * 4;
#define RD_OFF(mt, i) ((mt) * RT_BYTES + (i) * 8 * RT_STRIDE)
#define ST_ROW(mt, i) ((mt) * 32 + (i) * 8)
    v4f part[2][4][4];
#pragma unroll
    for (int mt = 0; mt < 2; ++mt)
#pragma unroll
      for (int i = 0; i < 4; ++i)
#pragma unroll
        for (int w2 = 0; w2 < 4; ++w2) part[mt][i][w2] = *(const LDSP v4f*)(uintptr_t)(rbase + w2 * WV_BYTES + RD_OFF(mt, i));
    FENCE();
    const int anyb = braw[0] | braw[1] | braw[2] | braw[3];
    const float scl = anyb ? __builtin_nanf("") : sc;
#pragma unroll
    for (int mt = 0; mt < 2; ++mt)
#pragma unroll
      for (int i = 0; i < 4; ++i) {
        v4f sum = {};
#pragma unroll
        for (int w2 = 0; w2 < 4; ++w2)
          sum += part[mt][i][w2];
        v4f o;
#pragma unroll
        for (int e = 0; e < 4; ++e) { const float y = sum[e] * scl; o[e] = anyb ? y : fmaxf(y, 0.0f); }
        asm volatile("global_store_dwordx4 %0, %1, off sc0 sc1\n\ts_nop 1" :: "v"(op + (size_t)ST_ROW(mt, i) * NDIM), "v"(o) : "memory");
      }
  }
  if (tid == 0) __hip_atomic_store(ctr, call, __ATOMIC_RELAXED, __HIP_MEMORY_SCOPE_AGENT);
  STAMP(5);
}

extern "C" void kernel_launch(int rank, int nranks, ncclComm_t comm,
                              hipStream_t stream,
                              void* const* d_in, const int* in_sizes, int n_in,
                              void* d_out, int out_size,
                              const int* in_dtypes, int out_dtype,
                              void* const* d_out_all,
                              void* const* d_ws_all, size_t ws_bytes) {
  (void)comm; (void)in_sizes; (void)n_in; (void)out_size; (void)in_dtypes; (void)out_dtype; (void)d_out_all;
  if (nranks != NR || ws_bytes < WS_NEED) { fprintf(stderr, "unexpected config\n"); return; }
  static bool attr_done = false;
  if (!attr_done) {
    HIP_CHECK(hipFuncSetAttribute((const void*)a2a_gemm_kernel, hipFuncAttributeMaxDynamicSharedMemorySize, LDS_BYTES));
    attr_done = true;
  }
  Params p;
  p.x = (const uint8_t*)d_in[0];
  p.w = (const uint8_t*)d_in[1];
  p.sx = (const float*)d_in[2];
  p.sw = (const float*)d_in[3];
  p.out = (float*)d_out;
  for (int i = 0; i < NR; ++i) p.ws[i] = (uint8_t*)d_ws_all[i];
  p.myws = (uint8_t*)d_ws_all[rank];
  p.rank = rank;
  p.pad = 0;
  a2a_gemm_kernel<<<NWG, NTHREADS, LDS_BYTES, stream>>>(p);
  HIP_CHECK(hipGetLastError());
}
```
